```python
import numpy as np

kernel_name = "gpu_ag_gemm_m8192_k8192_n8192_bf16_none"


ABSMAX_RTOL = 0.008

COMPUTE_DTYPE = "bfloat16"
EPILOGUE = "none"
IN_QUANT = None
OUT_QUANT = None
IN_DTYPES = {"x": "bfloat16", "w_mat": "bfloat16"}
WORLD_SIZE = 8

def _to_bf16(x):
    u = np.ascontiguousarray(x, dtype=np.float32).view(np.uint32)
    return ((u + 0x7FFF + ((u >> 16) & 1)) & 0xFFFF0000).view(np.float32)

def setup_inputs():

    x = _to_bf16(np.random.randn(8192, 8192).astype(np.float32))
    w_mat = _to_bf16((np.random.randn(8192, 8192) / np.sqrt(8192)).astype(np.float32))
    return {"x": x, "w_mat": w_mat}

def _epilogue(y):
    return y

def reference(x, w_mat):
    y = (x @ w_mat).astype(np.float32)
    return _epilogue(y)
```

```cpp
#include <hip/hip_runtime.h>
#include <hip/hip_bf16.h>
#include <rccl/rccl.h>
#include <cstdio>
#include <cstdint>

#define HIP_CHECK(expr) do { hipError_t _e = (expr); if (_e != hipSuccess) { \
    fprintf(stderr, "HIP error %s at %s:%d\n", hipGetErrorString(_e), __FILE__, __LINE__); \
    return; } } while (0)

using bf16 = __hip_bfloat16;
using bf16x8 = __attribute__((ext_vector_type(8))) short;
using f32x4 = __attribute__((ext_vector_type(4))) float;
typedef unsigned int u32;
typedef unsigned long long u64;

constexpr int NR = 8;
constexpr int MLOC = 1024;
constexpr int KK = 8192;
constexpr int NLOC = 1024;
constexpr int MTOT = MLOC * NR;
constexpr int KC = 256;
constexpr int NCH = KK / KC;
constexpr int RT = 256;
constexpr int NRT = MLOC / RT;
constexpr size_t CHUNK_ELEMS = (size_t)MLOC * KC;
constexpr size_t SHARD_BYTES = (size_t)MLOC * KK * 2;

constexpr size_t CTR_OFF = 0;
constexpr size_t TMO_OFF = 1024;
constexpr size_t SRCFLAG_OFF = 4096;
constexpr size_t LOCFLAG_OFF = 262144;
constexpr size_t STAGE_OFF = 1ull << 20;
constexpr size_t GATH_OFF = STAGE_OFF + 2 * SHARD_BYTES;
constexpr size_t WT_OFF = GATH_OFF + NR * SHARD_BYTES;
constexpr size_t WS_NEEDED = WT_OFF + SHARD_BYTES;

constexpr u64 DEADLINE_TICKS = 100000000ull;

__device__ __forceinline__ u32* flag_ptr(char* ws, size_t off, int p, int c, int j) {
  return (u32*)(ws + off + (size_t)(((p * NCH) + c) * NRT + j) * 128);
}

template <bool SYS>
__device__ __forceinline__ void wait_flag(u32* flag, u32 seq, u32* tmo) {
  u64 t0 = __builtin_amdgcn_s_memrealtime();
  while (true) {
    u32 v = SYS ? __hip_atomic_load(flag, __ATOMIC_RELAXED, __HIP_MEMORY_SCOPE_SYSTEM)
                : __hip_atomic_load(flag, __ATOMIC_RELAXED, __HIP_MEMORY_SCOPE_AGENT);
    if (v >= seq) break;
    if (__hip_atomic_load(tmo, __ATOMIC_RELAXED, __HIP_MEMORY_SCOPE_AGENT) == seq ||
        __builtin_amdgcn_s_memrealtime() - t0 > DEADLINE_TICKS) {
      __hip_atomic_store(tmo, seq, __ATOMIC_RELAXED, __HIP_MEMORY_SCOPE_AGENT);
      break;
    }
    __builtin_amdgcn_s_sleep(2);
  }
}

constexpr int BM = 256, BK = 64, HALF = 128, HT = HALF * BK;
constexpr int SHM_GEMM = 8 * HT * 2;
constexpr int SHM_SMALL = 64 * 72 * 2;
__device__ __forceinline__ int lds_byte(int r, int c) {
  int st = (r >> 4) * 2 + (c >> 5), rr = r & 15, cc = c & 31, ob = rr * 64 + cc * 2;
  return st * 1024 + (ob ^ (((ob >> 9) & 1) << 5));
}
__device__ __forceinline__ void stage_rc(int b, int& R, int& C) {
  int st = b / 1024, sb = b % 1024, swz = sb ^ (((sb >> 9) & 1) << 5);
  R = (st >> 1) * 16 + swz / 64; C = (st & 1) * 32 + (swz % 64) / 2;
}

__global__ __launch_bounds__(512, 2)
void fused_ag_gemm(const bf16* __restrict__ x, const bf16* __restrict__ w, bf16* __restrict__ out,
                   void* const* __restrict__ ws_all, int rank, int phase) {
  extern __shared__ __attribute__((aligned(16))) char smem[];
  char* ws = (char*)ws_all[rank];
  u32* tmo = (u32*)(ws + TMO_OFF);
  u32 call;
  {
    u32* sh = (u32*)(smem + (phase == 2 ? SHM_GEMM : SHM_SMALL));
    if (threadIdx.x == 0) {
      u32 t = __hip_atomic_fetch_add((u32*)(ws + CTR_OFF + phase * 128), 1u, __ATOMIC_RELAXED,
                                     __HIP_MEMORY_SCOPE_AGENT);
      sh[0] = t / gridDim.x + 1;
    }
    __syncthreads();
    call = __builtin_amdgcn_readfirstlane(sh[0]);
    __syncthreads();
  }
  const int par = call & 1;

  if (phase == 0) {
    const int b = blockIdx.x;
    if (b < NCH * NRT) {
      const int c = b / NRT, j = b % NRT;
      const uint4* src = (const uint4*)(x + (size_t)(RT * j) * KK + (size_t)c * KC);
      uint4* dst = (uint4*)(ws + STAGE_OFF + (size_t)par * SHARD_BYTES) +
                   ((size_t)c * CHUNK_ELEMS + (size_t)(RT * j) * KC) / 8;
      const int tr = threadIdx.x >> 5, tc = threadIdx.x & 31;
#pragma unroll 1
      for (int r0 = 0; r0 < RT; r0 += 32) {
        uint4 v[4];
#pragma unroll
        for (int u = 0; u < 4; ++u) v[u] = src[(size_t)(r0 + u * 8 + tr) * (KK / 8) + tc];
#pragma unroll
        for (int u = 0; u < 4; ++u) dst[(size_t)(r0 + u * 8 + tr) * (KC / 8) + tc] = v[u];
      }
      asm volatile("s_waitcnt vmcnt(0)" ::: "memory");
      __syncthreads();
      if (threadIdx.x < NR - 1) {
        __builtin_amdgcn_fence(__ATOMIC_RELEASE, "");
        asm volatile("s_waitcnt vmcnt(0)" ::: "memory");
        u32 p = (u32)(rank + 1 + threadIdx.x) % NR;
        if (p < (u32)NR) {
          u32* f = flag_ptr((char*)ws_all[p], SRCFLAG_OFF, rank, c, j);
          __hip_atomic_store(f, call, __ATOMIC_RELAXED, __HIP_MEMORY_SCOPE_SYSTEM);
        }
      }
    } else {
      const int idx = b - NCH * NRT;
      const u32 p = (u32)(rank + 1 + idx % (NR - 1)) % NR;
      const int j = idx / (NR - 1);
      if (p >= (u32)NR || j >= NRT) return;
      const char* pws = (const char*)ws_all[p];
#pragma unroll 1
      for (int c = 0; c < NCH; ++c) {
        if (threadIdx.x == 0) {
          wait_flag<true>(flag_ptr(ws, SRCFLAG_OFF, p, c, j), call, tmo);
          __builtin_amdgcn_fence(__ATOMIC_ACQUIRE, "");
        }
        __syncthreads();
        const size_t eoff = ((size_t)c * CHUNK_ELEMS + (size_t)(RT * j) * KC) / 8;
        const uint4* src = (const uint4*)(pws + STAGE_OFF + (size_t)par * SHARD_BYTES) + eoff;
        uint4* dst = (uint4*)(ws + GATH_OFF + (size_t)p * SHARD_BYTES) + eoff;
        constexpr int N16 = RT * KC / 8;
#pragma unroll 1
        for (int i0 = 0; i0 < N16; i0 += 1024) {
          uint4 v[4];
#pragma unroll
          for (int u = 0; u < 4; ++u) v[u] = src[i0 + u * 256 + threadIdx.x];
#pragma unroll
          for (int u = 0; u < 4; ++u) dst[i0 + u * 256 + threadIdx.x] = v[u];
        }
        asm volatile("s_waitcnt vmcnt(0)" ::: "memory");
        __syncthreads();
        if (threadIdx.x == 0) {
          __builtin_amdgcn_fence(__ATOMIC_RELEASE, "agent");
          asm volatile("s_waitcnt vmcnt(0)" ::: "memory");
          __hip_atomic_store(flag_ptr(ws, LOCFLAG_OFF, p, c, j), call, __ATOMIC_RELAXED,
                             __HIP_MEMORY_SCOPE_AGENT);
        }
      }
    }
    return;
  }

  if (phase == 1) {
    constexpr int TN = NLOC / 64;
    const int tk = blockIdx.x / TN, tn = blockIdx.x % TN;
    unsigned short* tile = (unsigned short*)smem;
    const int t = threadIdx.x;
#pragma unroll
    for (int i = 0; i < 2; ++i) {
      int k = (t >> 3) + i * 32, n8 = (t & 7) * 8;
      uint4 v = *(const uint4*)(w + (size_t)(tk * 64 + k) * NLOC + tn * 64 + n8);
      const unsigned short* e = (const unsigned short*)&v;
#pragma unroll
      for (int q = 0; q < 8; ++q) tile[(n8 + q) * 72 + k] = e[q];
    }
    __syncthreads();
    bf16* wT = (bf16*)(ws + WT_OFF);
#pragma unroll
    for (int i = 0; i < 2; ++i) {
      int n = (t >> 3) + i * 32, k8 = (t & 7) * 8;
      uint4 v = *(const uint4*)(tile + n * 72 + k8);
      *(uint4*)(wT + (size_t)(tn * 64 + n) * KK + tk * 64 + k8) = v;
    }
    return;
  }

  {
    bf16* shm = (bf16*)smem;
    const int wgid = blockIdx.x;
    const int xcd = wgid & 7, idx = wgid >> 3;
    const int pm = xcd * 4 + (idx >> 2);
    const int pn = idx & 3;
    const int brow = pm * BM, bcol = pn * BM;
    const u32 srcp = (u32)pm / NRT;
    const int jrt = pm % NRT;
    const bool remote = (srcp != (u32)rank);
    const bf16* A; size_t lda, cstride;
    if (remote) {
      A = (const bf16*)(ws + GATH_OFF + (size_t)srcp * SHARD_BYTES) + (size_t)(jrt * RT) * KC;
      lda = KC; cstride = CHUNK_ELEMS;
    } else {
      A = x + (size_t)(jrt * RT) * KK;
      lda = KK; cstride = KC;
    }
    const bf16* Bt = (const bf16*)(ws + WT_OFF) + (size_t)bcol * KK;
    u32* locf = flag_ptr(ws, LOCFLAG_OFF, srcp, 0, jrt);

#define SA(b, h) (shm + ((b) * 2 + (h)) * HT)
#define SB(b, h) (shm + (4 + (b) * 2 + (h)) * HT)
#define STAGE_A_X(T, P, half, kt) do { size_t _g = (size_t)(half) * HALF * lda + (size_t)((kt) >> 2) * cstride + (size_t)((kt) & 3) * BK; \
    for (int _i = 0; _i < 2; ++_i) { int _b = (T) * 16 + _i * 8192; int _r, _c; stage_rc(_b, _r, _c); \
      __builtin_amdgcn_global_load_lds((const unsigned*)(A + _g + (size_t)_r * lda + _c), \
        (unsigned*)((char*)(P) + _b), 16, 0, 0); } } while (0)
#define STAGE_B_X(T, P, half, kt) do { size_t _g = (size_t)(half) * HALF * KK + (size_t)(kt) * BK; \
    for (int _i = 0; _i < 2; ++_i) { int _b = (T) * 16 + _i * 8192; int _r, _c; stage_rc(_b, _r, _c); \
      __builtin_amdgcn_global_load_lds((const unsigned*)(Bt + _g + (size_t)_r * KK + _c), \
        (unsigned*)((char*)(P) + _b), 16, 0, 0); } } while (0)
#define STAGE_A(P, half, kt) STAGE_A_X(threadIdx.x, P, half, kt)
#define STAGE_B(P, half, kt) STAGE_B_X(threadIdx.x, P, half, kt)
#define LDA(dst, b, h) for (int m = 0; m < 4; ++m) for (int k = 0; k < 2; ++k) \
    dst[m][k] = *reinterpret_cast<const bf16x8*>((char*)SA(b, h) + lds_byte(wr * 64 + m * 16 + fr, k * 32 + fq * 8))
#define LDB(dst, b, h) for (int n = 0; n < 2; ++n) for (int k = 0; k < 2; ++k) \
    dst[n][k] = *reinterpret_cast<const bf16x8*>((char*)SB(b, h) + lds_byte(wc * 32 + n * 16 + fr, k * 32 + fq * 8))
#define MMA(ai, bj, At, Bq) do { __builtin_amdgcn_s_setprio(1); \
    for (int m = 0; m < 4; ++m) for (int n = 0; n < 2; ++n) for (int k = 0; k < 2; ++k) \
      acc[ai][bj][m][n] = __builtin_amdgcn_mfma_f32_16x16x32_bf16(At[m][k], Bq[n][k], acc[ai][bj][m][n], 0, 0, 0); \
    __builtin_amdgcn_s_setprio(0); } while (0)
#define WAIT_V(n) asm volatile("s_waitcnt vmcnt(" #n ")" ::: "memory")
#define WAIT_L(n) asm volatile("s_waitcnt lgkmcnt(" #n ")" ::: "memory")
#define BAR __builtin_amdgcn_s_barrier()
#define SCHED __builtin_amdgcn_sched_barrier(0)
#define WAITCHUNK(cidx) do { if (remote) { \
      wait_flag<false>((u32*)((char*)locf + (size_t)(cidx) * NRT * 128), call, tmo); \
      __builtin_amdgcn_fence(__ATOMIC_ACQUIRE, "agent"); } } while (0)

    int wid = threadIdx.x >> 6, lane = threadIdx.x & 63, wr = wid >> 2, wc = wid & 3, fr = lane & 15, fq = lane >> 4;
    f32x4 acc[2][2][4][2] = {};
    bf16x8 At[4][2], B0[2][2], B1[2][2];
    const int nt = KK / BK;
    WAITCHUNK(0);
    STAGE_B(SB(0, 0), 0, 0); STAGE_A(SA(0, 0), 0, 0);
    STAGE_B(SB(0, 1), 1, 0); STAGE_A(SA(0, 1), 1, 0);
    if (wr == 1) BAR;
    WAIT_V(4); BAR;
    STAGE_B(SB(1, 0), 0, 1); STAGE_A(SA(1, 0), 0, 1); STAGE_B(SB(1, 1), 1, 1);
    WAIT_V(6); BAR;
    for (int t = 0; t < nt - 2; t += 2) {
      if ((t & 3) == 2) WAITCHUNK((t + 2) >> 2);
      LDB(B0, 0, 0); SCHED; LDA(At, 0, 0); STAGE_A(SA(1, 1), 1, t + 1);
      WAIT_L(8); BAR; WAIT_L(0); MMA(0, 0, At, B0); BAR; SCHED;
      LDB(B1, 0, 1); STAGE_B(SB(0, 0), 0, t + 2);
      BAR; WAIT_L(0); MMA(0, 1, At, B1); BAR;
      LDA(At, 0, 1); STAGE_A(SA(0, 0), 0, t + 2);
      BAR; WAIT_L(0); MMA(1, 0, At, B0); BAR; SCHED;
      STAGE_B(SB(0, 1), 1, t + 2);
      WAIT_V(6); BAR; MMA(1, 1, At, B1); BAR;
      LDB(B0, 1, 0); SCHED; LDA(At, 1, 0); STAGE_A(SA(0, 1), 1, t + 2);
      WAIT_L(8); BAR; WAIT_L(0); MMA(0, 0, At, B0); BAR; SCHED;
      LDB(B1, 1, 1); STAGE_B(SB(1, 0), 0, t + 3);
      BAR; WAIT_L(0); MMA(0, 1, At, B1); BAR;
      LDA(At, 1, 1); STAGE_A(SA(1, 0), 0, t + 3);
      BAR; WAIT_L(0); MMA(1, 0, At, B0); BAR; SCHED;
      STAGE_B(SB(1, 1), 1, t + 3);
      WAIT_V(6); BAR; MMA(1, 1, At, B1); BAR;
    }
    int tz = threadIdx.x; asm volatile("" : "+v"(tz)); wid = tz >> 6; lane = tz & 63; wr = wid >> 2; wc = wid & 3; fr = lane & 15; fq = lane >> 4;
    { LDB(B0, 0, 0); WAIT_V(0); LDA(At, 0, 0); STAGE_A_X(tz, SA(1, 1), 1, nt - 1);
      BAR; WAIT_L(0); MMA(0, 0, At, B0); BAR;
      LDB(B1, 0, 1); BAR; WAIT_L(0); MMA(0, 1, At, B1); BAR;
      LDA(At, 0, 1); WAIT_V(4); BAR; WAIT_L(0); MMA(1, 0, At, B0); MMA(1, 1, At, B1); BAR; }
    { LDB(B0, 1, 0); LDA(At, 1, 0); WAIT_V(2); BAR; WAIT_L(0); MMA(0, 0, At, B0); BAR;
      LDB(B1, 1, 1); WAIT_V(0); BAR; WAIT_L(0); MMA(0, 1, At, B1); BAR;
      LDA(At, 1, 1); BAR; WAIT_L(0); MMA(1, 0, At, B0); MMA(1, 1, At, B1); BAR; }
    if (wr == 0) BAR;
    const bool bad = (__hip_atomic_load(tmo, __ATOMIC_RELAXED, __HIP_MEMORY_SCOPE_AGENT) == call);
    for (int ai = 0; ai < 2; ++ai) for (int bj = 0; bj < 2; ++bj) for (int m = 0; m < 4; ++m) for (int n = 0; n < 2; ++n) for (int jj = 0; jj < 4; ++jj) {
      float v = bad ? __builtin_nanf("") : acc[ai][bj][m][n][jj];
      out[(size_t)(brow + ai * HALF + wr * 64 + m * 16 + fq * 4 + jj) * NLOC + (bcol + bj * HALF + wc * 32 + n * 16 + fr)] = __float2bfloat16(v);
    }
  }
}

extern "C" void kernel_launch(int rank, int nranks, ncclComm_t comm,
                              hipStream_t stream,
                              void* const* d_in, const int* in_sizes, int n_in,
                              void* d_out, int out_size,
                              const int* in_dtypes, int out_dtype,
                              void* const* d_out_all,
                              void* const* d_ws_all, size_t ws_bytes) {
  static bool init = false;
  static hipStream_t side = nullptr;
  static hipEvent_t e_fork = nullptr, e_join = nullptr;
  if (nranks != NR || ws_bytes < WS_NEEDED) {
    fprintf(stderr, "unexpected config nranks=%d ws=%zu\n", nranks, ws_bytes);
    return;
  }
  if (!init) {
    HIP_CHECK(hipFuncSetAttribute((const void*)fused_ag_gemm, hipFuncAttributeMaxDynamicSharedMemorySize, SHM_GEMM + 64));
    HIP_CHECK(hipStreamCreateWithFlags(&side, hipStreamNonBlocking));
    HIP_CHECK(hipEventCreateWithFlags(&e_fork, hipEventDisableTiming));
    HIP_CHECK(hipEventCreateWithFlags(&e_join, hipEventDisableTiming));
    init = true;
  }
  const bf16* x = (const bf16*)d_in[0];
  const bf16* w = (const bf16*)d_in[1];
  bf16* out = (bf16*)d_out;
  const size_t shm = SHM_GEMM + 64;
  HIP_CHECK(hipEventRecord(e_fork, stream));
  HIP_CHECK(hipStreamWaitEvent(side, e_fork, 0));
  hipLaunchKernelGGL(fused_ag_gemm, dim3(NCH * NRT + (NR - 1) * NRT), dim3(256), SHM_SMALL + 64, stream,
                     x, w, out, d_ws_all, rank, 0);
  hipLaunchKernelGGL(fused_ag_gemm, dim3((KK / 64) * (NLOC / 64)), dim3(256), SHM_SMALL + 64, side,
                     x, w, out, d_ws_all, rank, 1);
  hipLaunchKernelGGL(fused_ag_gemm, dim3((MTOT / BM) * (NLOC / BM)), dim3(512), shm, side,
                     x, w, out, d_ws_all, rank, 2);
  HIP_CHECK(hipGetLastError());
  HIP_CHECK(hipEventRecord(e_join, side));
  HIP_CHECK(hipStreamWaitEvent(stream, e_join, 0));
}
```

```cpp
#include <hip/hip_runtime.h>
#include <hip/hip_bf16.h>
#include <rccl/rccl.h>
#include <cstdio>
#include <cstdint>

#define HIP_CHECK(expr) do { hipError_t _e = (expr); if (_e != hipSuccess) { \
    fprintf(stderr, "HIP error %s at %s:%d\n", hipGetErrorString(_e), __FILE__, __LINE__); \
    return; } } while (0)

using bf16 = __hip_bfloat16;
using bf16x8 = __attribute__((ext_vector_type(8))) short;
using f32x4 = __attribute__((ext_vector_type(4))) float;
typedef unsigned int u32;
typedef unsigned long long u64;
typedef __attribute__((ext_vector_type(4))) unsigned int u32x4;

constexpr int NR = 8;
constexpr int MLOC = 1024;
constexpr int KK = 8192;
constexpr int NLOC = 1024;
constexpr int MTOT = MLOC * NR;
constexpr int KC = 256;
constexpr int NCH = KK / KC;
constexpr int RT = 256;
constexpr int NRT = MLOC / RT;
constexpr size_t CHUNK_ELEMS = (size_t)MLOC * KC;
constexpr size_t SHARD_BYTES = (size_t)MLOC * KK * 2;

constexpr size_t CTR_OFF = 0;
constexpr size_t TMO_OFF = 1024;
constexpr size_t SRCFLAG_OFF = 4096;
constexpr size_t LOCFLAG_OFF = 262144;
constexpr size_t STAGE_OFF = 1ull << 20;
constexpr size_t GATH_OFF = STAGE_OFF + 2 * SHARD_BYTES;
constexpr size_t WT_OFF = GATH_OFF + NR * SHARD_BYTES;
constexpr size_t WS_NEEDED = WT_OFF + SHARD_BYTES;

constexpr u64 DEADLINE_TICKS = 100000000ull;

__device__ __forceinline__ u32* flag_ptr(char* ws, size_t off, int p, int c, int j) {
  return (u32*)(ws + off + (size_t)(((p * NCH) + c) * NRT + j) * 128);
}

template <bool SYS>
__device__ __forceinline__ void wait_flag(u32* flag, u32 target, u32 seq, u32* tmo) {
  u64 t0 = __builtin_amdgcn_s_memrealtime();
  while (true) {
    u32 v = SYS ? __hip_atomic_load(flag, __ATOMIC_RELAXED, __HIP_MEMORY_SCOPE_SYSTEM)
                : __hip_atomic_load(flag, __ATOMIC_RELAXED, __HIP_MEMORY_SCOPE_AGENT);
    if (v >= target) break;
    if (__hip_atomic_load(tmo, __ATOMIC_RELAXED, __HIP_MEMORY_SCOPE_AGENT) == seq ||
        __builtin_amdgcn_s_memrealtime() - t0 > DEADLINE_TICKS) {
      __hip_atomic_store(tmo, seq, __ATOMIC_RELAXED, __HIP_MEMORY_SCOPE_AGENT);
      break;
    }
    __builtin_amdgcn_s_sleep(SYS ? 2 : 127);
  }
}

#define STAMP(i) do {} while (0)

constexpr int BM = 256, BK = 64, HALF = 128, HT = HALF * BK;
constexpr int SHM_GEMM = 8 * HT * 2;
constexpr int SHM_SMALL = 64 * 72 * 2;
__device__ __forceinline__ int lds_byte(int r, int c) {
  int st = (r >> 4) * 2 + (c >> 5), rr = r & 15, cc = c & 31, ob = rr * 64 + cc * 2;
  return st * 1024 + (ob ^ (((ob >> 9) & 1) << 5));
}
__device__ __forceinline__ void stage_rc(int b, int& R, int& C) {
  int st = b / 1024, sb = b % 1024, swz = sb ^ (((sb >> 9) & 1) << 5);
  R = (st >> 1) * 16 + swz / 64; C = (st & 1) * 32 + (swz % 64) / 2;
}

__global__ __launch_bounds__(512, 2)
void fused_ag_gemm(const bf16* __restrict__ x, const bf16* __restrict__ w, bf16* __restrict__ out,
                   void* const* __restrict__ ws_all, int rank, int phase) {
  extern __shared__ __attribute__((aligned(16))) char smem[];
  char* ws = (char*)ws_all[rank];
  u32* tmo = (u32*)(ws + TMO_OFF);
  u32 call;
  {
    u32* sh = (u32*)(smem + (phase == 2 ? SHM_GEMM : SHM_SMALL));
    if (threadIdx.x == 0) {
      u32 t = __hip_atomic_fetch_add((u32*)(ws + CTR_OFF + phase * 128), 1u, __ATOMIC_RELAXED,
                                     __HIP_MEMORY_SCOPE_AGENT);
      sh[0] = t / gridDim.x + 1;
    }
    __syncthreads();
    call = __builtin_amdgcn_readfirstlane(sh[0]);
    __syncthreads();
  }
  const int par = call & 1;

  if (phase == 0) {
    const int b = blockIdx.x;
    if (threadIdx.x == 0) STAMP(b);
    if (b < NCH * NRT) {
      const int c = b / NRT, j = b % NRT;
      const uint4* src = (const uint4*)(x + (size_t)(RT * j) * KK + (size_t)c * KC);
      uint4* dst = (uint4*)(ws + STAGE_OFF + (size_t)par * SHARD_BYTES) +
                   ((size_t)c * CHUNK_ELEMS + (size_t)(RT * j) * KC) / 8;
      const int tr = threadIdx.x >> 5, tc = threadIdx.x & 31;
#pragma unroll 1
      for (int r0 = 0; r0 < RT; r0 += 32) {
        uint4 v[4];
#pragma unroll
        for (int u = 0; u < 4; ++u) v[u] = src[(size_t)(r0 + u * 8 + tr) * (KK / 8) + tc];
#pragma unroll
        for (int u = 0; u < 4; ++u) dst[(size_t)(r0 + u * 8 + tr) * (KC / 8) + tc] = v[u];
      }
      asm volatile("s_waitcnt vmcnt(0)" ::: "memory");
      __syncthreads();
      if (threadIdx.x < NR - 1) {
        __builtin_amdgcn_fence(__ATOMIC_RELEASE, "");
        asm volatile("s_waitcnt vmcnt(0)" ::: "memory");
        u32 p = (u32)(rank + 1 + threadIdx.x) % NR;
        if (p < (u32)NR) {
          u32* f = flag_ptr((char*)ws_all[p], SRCFLAG_OFF, rank, c, j);
          __hip_atomic_store(f, call, __ATOMIC_RELAXED, __HIP_MEMORY_SCOPE_SYSTEM);
        }
        if (threadIdx.x == 0) STAMP(256 + b);
      }
    } else {
      const int idx = b - NCH * NRT;
      const u32 p = (u32)(rank + 1 + idx % (NR - 1)) % NR;
      const int j = idx / (NR - 1);
      if (p >= (u32)NR || j >= NRT) return;
      const char* pws = (const char*)ws_all[p];
      constexpr int C0 = 4;
      const size_t joff = (size_t)(RT * j) * KC / 8;
      const uint4* src0 = (const uint4*)(pws + STAGE_OFF + (size_t)par * SHARD_BYTES) + joff;
      uint4* dst0 = (uint4*)(ws + GATH_OFF + (size_t)p * SHARD_BYTES) + joff;
      constexpr int N16 = RT * KC / 8;
      constexpr int ITC = N16 / 1024;
#pragma unroll 1
      for (int it = 0; it < NCH * ITC; ++it) {
        const int c = it / ITC, i0 = (it % ITC) * 1024;
        if (i0 == 0 && (c == 0 || c == C0)) {
          if (threadIdx.x < 64) {
            const int cc = threadIdx.x;
            if (cc >= c && cc < (c == 0 ? C0 : NCH))
              wait_flag<true>(flag_ptr(ws, SRCFLAG_OFF, p, cc, j), call, call, tmo);
            __builtin_amdgcn_fence(__ATOMIC_ACQUIRE, "");
            asm volatile("s_waitcnt vmcnt(0)" ::: "memory");
          }
          __syncthreads();
        }
        const size_t eoff = (size_t)c * (CHUNK_ELEMS / 8) + i0;
        u32x4 v[4];
#pragma unroll
        for (int u = 0; u < 4; ++u) v[u] = *(const u32x4*)(src0 + eoff + u * 256 + threadIdx.x);
#pragma unroll
        for (int u = 0; u < 4; ++u) {
          uint4* d = dst0 + eoff + u * 256 + threadIdx.x;
          asm volatile("global_store_dwordx4 %0, %1, off sc1\n\ts_nop 1" :: "v"(d), "v"(v[u]) : "memory");
        }
        if (i0 == 0 && c > 0) {
          asm volatile("s_waitcnt vmcnt(4)" ::: "memory");
          if ((threadIdx.x & 63) == 0) {
            __hip_atomic_fetch_add(flag_ptr(ws, LOCFLAG_OFF, p, c - 1, j), 1u, __ATOMIC_RELAXED,
                                   __HIP_MEMORY_SCOPE_AGENT);
            if (threadIdx.x == 0) STAMP(1024 + idx * 32 + c - 1);
          }
        }
      }
      asm volatile("s_waitcnt vmcnt(0)" ::: "memory");
      if ((threadIdx.x & 63) == 0) {
        __hip_atomic_fetch_add(flag_ptr(ws, LOCFLAG_OFF, p, NCH - 1, j), 1u, __ATOMIC_RELAXED,
                               __HIP_MEMORY_SCOPE_AGENT);
        if (threadIdx.x == 0) STAMP(1024 + idx * 32 + NCH - 1);
      }
    }
    return;
  }

  if (phase == 1) {
    constexpr int TN = NLOC / 64;
    const int tk = blockIdx.x / TN, tn = blockIdx.x % TN;
    if (threadIdx.x == 0 && (blockIdx.x == 0 || blockIdx.x == gridDim.x - 1)) STAMP(3000 + (blockIdx.x != 0));
    unsigned short* tile = (unsigned short*)smem;
    const int t = threadIdx.x;
#pragma unroll
    for (int i = 0; i < 2; ++i) {
      int k = (t >> 3) + i * 32, n8 = (t & 7) * 8;
      uint4 v = *(const uint4*)(w + (size_t)(tk * 64 + k) * NLOC + tn * 64 + n8);
      const unsigned short* e = (const unsigned short*)&v;
#pragma unroll
      for (int q = 0; q < 8; ++q) tile[(n8 + q) * 72 + k] = e[q];
    }
    __syncthreads();
    bf16* wT = (bf16*)(ws + WT_OFF);
#pragma unroll
    for (int i = 0; i < 2; ++i) {
      int n = (t >> 3) + i * 32, k8 = (t & 7) * 8;
      uint4 v = *(const uint4*)(tile + n * 72 + k8);
      *(uint4*)(wT + (size_t)(tn * 64 + n) * KK + tk * 64 + k8) = v;
    }
    return;
  }

  {
    bf16* shm = (bf16*)smem;
    const int wgid = blockIdx.x;
    const int xcd = wgid & 7, idx = wgid >> 3;
    const int pm = xcd * 4 + (idx >> 2);
    const int pn = idx & 3;
    const int brow = pm * BM, bcol = pn * BM;
    const u32 srcp = (u32)pm / NRT;
    const int jrt = pm % NRT;
    const bool remote = (srcp != (u32)rank);
    const bf16* A; size_t lda, cstride;
    if (remote) {
      A = (const bf16*)(ws + GATH_OFF + (size_t)srcp * SHARD_BYTES) + (size_t)(jrt * RT) * KC;
      lda = KC; cstride = CHUNK_ELEMS;
    } else {
      A = x + (size_t)(jrt * RT) * KK;
      lda = KK; cstride = KC;
    }
    const bf16* Bt = (const bf16*)(ws + WT_OFF) + (size_t)bcol * KK;
    u32* locf = flag_ptr(ws, LOCFLAG_OFF, srcp, 0, jrt);

#define SA(b, h) (shm + ((b) * 2 + (h)) * HT)
#define SB(b, h) (shm + (4 + (b) * 2 + (h)) * HT)
#define STAGE_A_X(T, P, half, kt) do { size_t _g = (size_t)(half) * HALF * lda + (size_t)((kt) >> 2) * cstride + (size_t)((kt) & 3) * BK; \
    for (int _i = 0; _i < 2; ++_i) { int _b = (T) * 16 + _i * 8192; int _r, _c; stage_rc(_b, _r, _c); \
      __builtin_amdgcn_global_load_lds((const unsigned*)(A + _g + (size_t)_r * lda + _c), \
        (unsigned*)((char*)(P) + _b), 16, 0, 16); } } while (0)
#define STAGE_B_X(T, P, half, kt) do { size_t _g = (size_t)(half) * HALF * KK + (size_t)(kt) * BK; \
    for (int _i = 0; _i < 2; ++_i) { int _b = (T) * 16 + _i * 8192; int _r, _c; stage_rc(_b, _r, _c); \
      __builtin_amdgcn_global_load_lds((const unsigned*)(Bt + _g + (size_t)_r * KK + _c), \
        (unsigned*)((char*)(P) + _b), 16, 0, 0); } } while (0)
#define STAGE_A(P, half, kt) STAGE_A_X(threadIdx.x, P, half, kt)
#define STAGE_B(P, half, kt) STAGE_B_X(threadIdx.x, P, half, kt)
#define LDA(dst, b, h) for (int m = 0; m < 4; ++m) for (int k = 0; k < 2; ++k) \
    dst[m][k] = *reinterpret_cast<const bf16x8*>((char*)SA(b, h) + lds_byte(wr * 64 + m * 16 + fr, k * 32 + fq * 8))
#define LDB(dst, b, h) for (int n = 0; n < 2; ++n) for (int k = 0; k < 2; ++k) \
    dst[n][k] = *reinterpret_cast<const bf16x8*>((char*)SB(b, h) + lds_byte(wc * 32 + n * 16 + fr, k * 32 + fq * 8))
#define MMA(ai, bj, At, Bq) do { __builtin_amdgcn_s_setprio(1); \
    for (int m = 0; m < 4; ++m) for (int n = 0; n < 2; ++n) for (int k = 0; k < 2; ++k) \
      acc[ai][bj][m][n] = __builtin_amdgcn_mfma_f32_16x16x32_bf16(At[m][k], Bq[n][k], acc[ai][bj][m][n], 0, 0, 0); \
    __builtin_amdgcn_s_setprio(0); } while (0)
#define WAIT_V(n) asm volatile("s_waitcnt vmcnt(" #n ")" ::: "memory")
#define WAIT_L(n) asm volatile("s_waitcnt lgkmcnt(" #n ")" ::: "memory")
#define BAR __builtin_amdgcn_s_barrier()
#define SCHED __builtin_amdgcn_sched_barrier(0)
#define WAITCHUNK(cidx) do { if (remote) { \
      wait_flag<false>((u32*)((char*)locf + (size_t)(cidx) * NRT * 128), call * 4u, call, tmo); \
      asm volatile("" ::: "memory"); } } while (0)

    int wid = threadIdx.x >> 6, lane = threadIdx.x & 63, wr = wid >> 2, wc = wid & 3, fr = lane & 15, fq = lane >> 4;
    f32x4 acc[2][2][4][2] = {};
    bf16x8 At[4][2], B0[2][2], B1[2][2];
    const int nt = KK / BK;
    if (threadIdx.x == 0) STAMP(2048 + wgid);
    WAITCHUNK(0);
    STAGE_B(SB(0, 0), 0, 0); STAGE_A(SA(0, 0), 0, 0);
    STAGE_B(SB(0, 1), 1, 0); STAGE_A(SA(0, 1), 1, 0);
    if (wr == 1) BAR;
    WAIT_V(4); BAR;
    STAGE_B(SB(1, 0), 0, 1); STAGE_A(SA(1, 0), 0, 1); STAGE_B(SB(1, 1), 1, 1);
    WAIT_V(6); BAR;
    for (int t = 0; t < nt - 2; t += 2) {
      if ((t & 3) == 2) { WAITCHUNK((t + 2) >> 2); if (threadIdx.x == 0) STAMP(4096 + wgid * 32 + ((t + 2) >> 2)); }
      LDB(B0, 0, 0); SCHED; LDA(At, 0, 0); STAGE_A(SA(1, 1), 1, t + 1);
      WAIT_L(8); BAR; WAIT_L(0); MMA(0, 0, At, B0); BAR; SCHED;
      LDB(B1, 0, 1); STAGE_B(SB(0, 0), 0, t + 2);
      BAR; WAIT_L(0); MMA(0, 1, At, B1); BAR;
      LDA(At, 0, 1); STAGE_A(SA(0, 0), 0, t + 2);
      BAR; WAIT_L(0); MMA(1, 0, At, B0); BAR; SCHED;
      STAGE_B(SB(0, 1), 1, t + 2);
      WAIT_V(6); BAR; MMA(1, 1, At, B1); BAR;
      LDB(B0, 1, 0); SCHED; LDA(At, 1, 0); STAGE_A(SA(0, 1), 1, t + 2);
      WAIT_L(8); BAR; WAIT_L(0); MMA(0, 0, At, B0); BAR; SCHED;
      LDB(B1, 1, 1); STAGE_B(SB(1, 0), 0, t + 3);
      BAR; WAIT_L(0); MMA(0, 1, At, B1); BAR;
      LDA(At, 1, 1); STAGE_A(SA(1, 0), 0, t + 3);
      BAR; WAIT_L(0); MMA(1, 0, At, B0); BAR; SCHED;
      STAGE_B(SB(1, 1), 1, t + 3);
      WAIT_V(6); BAR; MMA(1, 1, At, B1); BAR;
    }
    int tz = threadIdx.x; asm volatile("" : "+v"(tz)); wid = tz >> 6; lane = tz & 63; wr = wid >> 2; wc = wid & 3; fr = lane & 15; fq = lane >> 4;
    { LDB(B0, 0, 0); WAIT_V(0); LDA(At, 0, 0); STAGE_A_X(tz, SA(1, 1), 1, nt - 1);
      BAR; WAIT_L(0); MMA(0, 0, At, B0); BAR;
      LDB(B1, 0, 1); BAR; WAIT_L(0); MMA(0, 1, At, B1); BAR;
      LDA(At, 0, 1); WAIT_V(4); BAR; WAIT_L(0); MMA(1, 0, At, B0); MMA(1, 1, At, B1); BAR; }
    { LDB(B0, 1, 0); LDA(At, 1, 0); WAIT_V(2); BAR; WAIT_L(0); MMA(0, 0, At, B0); BAR;
      LDB(B1, 1, 1); WAIT_V(0); BAR; WAIT_L(0); MMA(0, 1, At, B1); BAR;
      LDA(At, 1, 1); BAR; WAIT_L(0); MMA(1, 0, At, B0); MMA(1, 1, At, B1); BAR; }
    if (wr == 0) BAR;
    if (threadIdx.x == 0) STAMP(2304 + wgid);
    const bool bad = (__hip_atomic_load(tmo, __ATOMIC_RELAXED, __HIP_MEMORY_SCOPE_AGENT) == call);
    for (int ai = 0; ai < 2; ++ai) for (int bj = 0; bj < 2; ++bj) for (int m = 0; m < 4; ++m) for (int n = 0; n < 2; ++n) for (int jj = 0; jj < 4; ++jj) {
      float v = bad ? __builtin_nanf("") : acc[ai][bj][m][n][jj];
      out[(size_t)(brow + ai * HALF + wr * 64 + m * 16 + fq * 4 + jj) * NLOC + (bcol + bj * HALF + wc * 32 + n * 16 + fr)] = __float2bfloat16(v);
    }
  }
}

extern "C" void kernel_launch(int rank, int nranks, ncclComm_t comm,
                              hipStream_t stream,
                              void* const* d_in, const int* in_sizes, int n_in,
                              void* d_out, int out_size,
                              const int* in_dtypes, int out_dtype,
                              void* const* d_out_all,
                              void* const* d_ws_all, size_t ws_bytes) {
  static bool init_a[NR] = {};
  static hipStream_t side_a[NR] = {};
  static hipEvent_t e_fork_a[NR] = {}, e_join_a[NR] = {};
  if (nranks != NR || ws_bytes < WS_NEEDED) {
    fprintf(stderr, "unexpected config nranks=%d ws=%zu\n", nranks, ws_bytes);
    return;
  }
  if (rank < 0 || rank >= NR) return;
  bool& init = init_a[rank];
  hipStream_t& side = side_a[rank];
  hipEvent_t& e_fork = e_fork_a[rank];
  hipEvent_t& e_join = e_join_a[rank];
  if (!init) {
    HIP_CHECK(hipFuncSetAttribute((const void*)fused_ag_gemm, hipFuncAttributeMaxDynamicSharedMemorySize, SHM_GEMM + 64));
    HIP_CHECK(hipStreamCreateWithFlags(&side, hipStreamNonBlocking));
    HIP_CHECK(hipEventCreateWithFlags(&e_fork, hipEventDisableTiming));
    HIP_CHECK(hipEventCreateWithFlags(&e_join, hipEventDisableTiming));
    init = true;
  }
  const bf16* x = (const bf16*)d_in[0];
  const bf16* w = (const bf16*)d_in[1];
  bf16* out = (bf16*)d_out;
  const size_t shm = SHM_GEMM + 64;
  HIP_CHECK(hipEventRecord(e_fork, stream));
  HIP_CHECK(hipStreamWaitEvent(side, e_fork, 0));
  hipLaunchKernelGGL(fused_ag_gemm, dim3(NCH * NRT + (NR - 1) * NRT), dim3(256), SHM_SMALL + 64, stream,
                     x, w, out, d_ws_all, rank, 0);
  hipLaunchKernelGGL(fused_ag_gemm, dim3((KK / 64) * (NLOC / 64)), dim3(256), SHM_SMALL + 64, side,
                     x, w, out, d_ws_all, rank, 1);
  hipLaunchKernelGGL(fused_ag_gemm, dim3((MTOT / BM) * (NLOC / BM)), dim3(512), shm, side,
                     x, w, out, d_ws_all, rank, 2);
  HIP_CHECK(hipGetLastError());
  HIP_CHECK(hipEventRecord(e_join, side));
  HIP_CHECK(hipStreamWaitEvent(stream, e_join, 0));
}
```
